```python
import numpy as np

kernel_name = "gpu_gemm_rs_m4096_k4096_n4096_e4m3e4m3_dequant"


ABSMAX_RTOL = 0.03

COMPUTE_DTYPE = "float32"
EPILOGUE = "dequant"
IN_QUANT = None
OUT_QUANT = None
IN_FP8 = "e4m3e4m3"
IN_DTYPES = {"x": "e4m3", "w_mat": "e4m3",
             "scale_x": "float32", "scale_w": "float32"}
WORLD_SIZE = 8

def _snap_fp8(x, mant_bits, max_abs, min_normal):
    a = np.abs(x.astype(np.float64))
    _m, _e = np.frexp(a)
    q = np.where(a >= min_normal, np.ldexp(1.0, _e - 1 - mant_bits),
                 min_normal * 2.0 ** -mant_bits)
    snapped = np.minimum(np.round(a / q) * q, max_abs)
    return (np.sign(x) * snapped).astype(np.float32)

def setup_inputs():

    x = _snap_fp8(np.random.randn(4096, 4096).astype(np.float32), 3, 448.0, 2.0 ** -6)
    w_mat = _snap_fp8(np.random.randn(4096, 4096).astype(np.float32), 3, 448.0, 2.0 ** -6)
    scale_x = (0.02 * (1.0 + np.random.rand(1))).astype(np.float32)
    scale_w = (0.02 * (1.0 + np.random.rand(1))).astype(np.float32)
    return {"x": x, "w_mat": w_mat, "scale_x": scale_x, "scale_w": scale_w}

def reference(x, w_mat, scale_x, scale_w):
    acc = (x @ w_mat).astype(np.float32)
    y = (acc * (scale_x * scale_w)).astype(np.float32)
    return y
```

```cpp
#include <hip/hip_runtime.h>
#include <rccl/rccl.h>
#include <cstdio>
#include <cstdint>

typedef __attribute__((ext_vector_type(4))) unsigned u32x4;
typedef __attribute__((ext_vector_type(8))) int i32x8;
typedef __attribute__((ext_vector_type(16))) float f32x16;
typedef __attribute__((ext_vector_type(4))) float f32x4;

namespace {

constexpr int NR = 8;
constexpr int GM = 4096, GK = 4096, GN = 4096;
constexpr int MS = GM / NR;
constexpr int KS = GK / NR;

constexpr unsigned CTR_OFF   = 0;
constexpr unsigned TMO_OFF   = 4096;
constexpr unsigned FLAGA_OFF = 8192;
constexpr unsigned FLAGB_OFF = 16384;
constexpr unsigned STAGE_OFF = 1u << 20;
constexpr unsigned B_BYTES   = KS * GN;
constexpr unsigned A_BYTES   = MS * KS;
constexpr unsigned SRC_STRIDE = B_BYTES + A_BYTES;
constexpr unsigned PAR_STRIDE = NR * SRC_STRIDE;
constexpr unsigned WS_NEEDED  = STAGE_OFF + 2 * PAR_STRIDE;

constexpr int AUX_SYS = 17;
constexpr unsigned long long DEADLINE_TICKS = 100000000ull;

struct Params {
  const unsigned char* x;
  const unsigned char* w;
  const float* sx;
  const float* sw;
  float* out;
  unsigned char* ws[NR];
  int rank;
};

__device__ __forceinline__ __amdgpu_buffer_rsrc_t make_rsrc(const void* p) {
  return __builtin_amdgcn_make_buffer_rsrc(const_cast<void*>(p), 0, WS_NEEDED, 0x00020000);
}

__device__ __forceinline__ void tr4x4(unsigned d0, unsigned d1, unsigned d2, unsigned d3,
                                      unsigned& t0, unsigned& t1, unsigned& t2, unsigned& t3) {
  const unsigned a01l = __builtin_amdgcn_perm(d1, d0, 0x05010400u);
  const unsigned a01h = __builtin_amdgcn_perm(d1, d0, 0x07030602u);
  const unsigned a23l = __builtin_amdgcn_perm(d3, d2, 0x05010400u);
  const unsigned a23h = __builtin_amdgcn_perm(d3, d2, 0x07030602u);
  t0 = __builtin_amdgcn_perm(a23l, a01l, 0x05040100u);
  t1 = __builtin_amdgcn_perm(a23l, a01l, 0x07060302u);
  t2 = __builtin_amdgcn_perm(a23h, a01h, 0x05040100u);
  t3 = __builtin_amdgcn_perm(a23h, a01h, 0x07060302u);
}

__device__ __forceinline__ i32x8 cat8(u32x4 lo, u32x4 hi) {
  i32x8 r;
  r[0] = lo[0]; r[1] = lo[1]; r[2] = lo[2]; r[3] = lo[3];
  r[4] = hi[0]; r[5] = hi[1]; r[6] = hi[2]; r[7] = hi[3];
  return r;
}

template <int U>
__device__ __forceinline__ void consume(unsigned long long& ready, f32x16 (&acc)[4],
                                        __amdgpu_buffer_rsrc_t rs, unsigned abase, unsigned bbase, int lane) {
  u32x4 af[U][2], bf[U][8];
#pragma unroll
  for (int t = 0; t < U; ++t) {
    const unsigned u = (unsigned)__builtin_ctzll(ready);
    ready &= ready - 1ull;
    const unsigned so = (u >> 3) * SRC_STRIDE, kc = u & 7u;
    const unsigned ua = __builtin_amdgcn_readfirstlane(abase + so + kc * 4096u);
    const unsigned ub = __builtin_amdgcn_readfirstlane(bbase + so + kc * 8192u);
    af[t][0] = __builtin_amdgcn_raw_buffer_load_b128(rs, lane * 16, ua, AUX_SYS);
    af[t][1] = __builtin_amdgcn_raw_buffer_load_b128(rs, lane * 16 + 1024, ua, AUX_SYS);
#pragma unroll
    for (int q = 0; q < 8; ++q)
      bf[t][q] = __builtin_amdgcn_raw_buffer_load_b128(rs, lane * 16 + (q & 3) * 1024, ub + (q >> 2) * 4096u, AUX_SYS);
  }
  __builtin_amdgcn_sched_barrier(0);
#pragma unroll
  for (int t = 0; t < U; ++t) {
    const i32x8 a8 = cat8(af[t][0], af[t][1]);
#pragma unroll
    for (int c = 0; c < 4; ++c) {
      const i32x8 b8 = cat8(bf[t][2 * c], bf[t][2 * c + 1]);
      acc[c] = __builtin_amdgcn_mfma_scale_f32_32x32x64_f8f6f4(a8, b8, acc[c], 0, 0, 0, 0x7f7f7f7f, 0, 0x7f7f7f7f);
    }
  }
}

}

__global__ void __launch_bounds__(256, 1) gemm_ag_fused(Params P) {
  const int lane = threadIdx.x & 63;
  const int wave = __builtin_amdgcn_readfirstlane(threadIdx.x >> 6);
  const int g = blockIdx.x & 7;
  const int wi = blockIdx.x >> 3;
  unsigned char* myws = P.ws[P.rank];

  unsigned* ctr = reinterpret_cast<unsigned*>(myws + CTR_OFF) + (blockIdx.x * 4 + wave);
  const unsigned ctr_v = __hip_atomic_load(ctr, __ATOMIC_RELAXED, __HIP_MEMORY_SCOPE_AGENT);

  if (wave >= 2) {
    const int pw = wave - 2;
    const unsigned p = (unsigned)(P.rank + 1 + g) & 7u;
    unsigned char* dws = P.ws[p];
    const __amdgpu_buffer_rsrc_t drs = make_rsrc(dws);

    const int a = wi + 32 * pw;
    const int akc = a >> 3, ai = a & 7;
    u32x4 av[4];
    {
      const unsigned char* xs = P.x + (size_t)(p * MS + ai * 64 + (lane & 31)) * KS + akc * 64 + (lane >> 5) * 32;
#pragma unroll
      for (int mb = 0; mb < 2; ++mb)
#pragma unroll
        for (int h = 0; h < 2; ++h)
          av[mb * 2 + h] = *reinterpret_cast<const u32x4*>(xs + (size_t)mb * 32 * KS + h * 16);
    }
    unsigned d[4][32];
    {
      const unsigned char* wsrc = P.w + (size_t)((lane >> 5) * 32) * GN + wi * 128 + (lane & 31) * 4;
#pragma unroll
      for (int t = 0; t < 4; ++t)
#pragma unroll
        for (int r = 0; r < 32; ++r)
          d[t][r] = *reinterpret_cast<const unsigned*>(wsrc + (size_t)((pw + 2 * t) * 64 + r) * GN);
    }
    asm volatile("" ::: "memory");
    const unsigned call = __builtin_amdgcn_readfirstlane(ctr_v) + 1u;
    if (lane == 0) __hip_atomic_store(ctr, call, __ATOMIC_RELAXED, __HIP_MEMORY_SCOPE_AGENT);
    const unsigned sbase = STAGE_OFF + (call & 1u) * PAR_STRIDE + (unsigned)P.rank * SRC_STRIDE;
    {
      const unsigned ub = sbase + B_BYTES + (unsigned)(ai * 8 + akc) * 4096u;
#pragma unroll
      for (int q = 0; q < 4; ++q)
        __builtin_amdgcn_raw_buffer_store_b128(av[q], drs, lane * 16 + q * 1024, ub, AUX_SYS);
    }
    unsigned* fa = reinterpret_cast<unsigned*>(dws + FLAGA_OFF) + (ai * 64 + P.rank * 8 + akc);
    unsigned* fb = reinterpret_cast<unsigned*>(dws + FLAGB_OFF) + (wi * 64 + P.rank * 8 + pw);
#pragma unroll
    for (int t = 0; t < 4; ++t) {
      unsigned tt[4][8];
#pragma unroll
      for (int q = 0; q < 8; ++q)
        tr4x4(d[t][4 * q], d[t][4 * q + 1], d[t][4 * q + 2], d[t][4 * q + 3],
              tt[0][q], tt[1][q], tt[2][q], tt[3][q]);
      const unsigned ub = sbase + (unsigned)(wi * 8 + pw + 2 * t) * 8192u;
      asm volatile("" ::: "memory");
#pragma unroll
      for (int c = 0; c < 4; ++c)
#pragma unroll
        for (int h = 0; h < 2; ++h) {
          u32x4 v = {tt[c][4 * h], tt[c][4 * h + 1], tt[c][4 * h + 2], tt[c][4 * h + 3]};
          __builtin_amdgcn_raw_buffer_store_b128(v, drs, lane * 16 + (c & 1) * 2048 + h * 1024,
                                                 ub + (c >> 1) * 4096, AUX_SYS);
        }
      if (t == 0) {
        asm volatile("s_waitcnt vmcnt(8)" ::: "memory");
        if (lane == 0) __hip_atomic_store(fa, call, __ATOMIC_RELAXED, __HIP_MEMORY_SCOPE_SYSTEM);
      } else {
        asm volatile("s_waitcnt vmcnt(9)" ::: "memory");
        if (lane == 0) __hip_atomic_store(fb + 2 * (t - 1), call, __ATOMIC_RELAXED, __HIP_MEMORY_SCOPE_SYSTEM);
      }
      asm volatile("" ::: "memory");
    }
    asm volatile("s_waitcnt vmcnt(0)" ::: "memory");
    if (lane == 0) __hip_atomic_store(fb + 6, call, __ATOMIC_RELAXED, __HIP_MEMORY_SCOPE_SYSTEM);
    return;
  }

  const unsigned call = __builtin_amdgcn_readfirstlane(ctr_v) + 1u;
  if (lane == 0) __hip_atomic_store(ctr, call, __ATOMIC_RELAXED, __HIP_MEMORY_SCOPE_AGENT);
  const unsigned par_off = STAGE_OFF + (call & 1u) * PAR_STRIDE;
  const int nj = 4 * g + (wi & 3);
  const int mi = wi >> 2;
  const int mb = wave;
  const __amdgpu_buffer_rsrc_t rs = make_rsrc(myws);
  const unsigned* fa = reinterpret_cast<const unsigned*>(myws + FLAGA_OFF) + mi * 64 + lane;
  const unsigned* fb = reinterpret_cast<const unsigned*>(myws + FLAGB_OFF) + nj * 64 + lane;
  unsigned* tmo = reinterpret_cast<unsigned*>(myws + TMO_OFF);

  f32x16 acc[4];
#pragma unroll
  for (int c = 0; c < 4; ++c)
#pragma unroll
    for (int r = 0; r < 16; ++r) acc[c][r] = 0.f;

  unsigned long long done = 0ull;
  const unsigned long long t0 = __builtin_amdgcn_s_memrealtime();
  bool failed = false;
  const unsigned abase = par_off + B_BYTES + (unsigned)(mi * 8) * 4096u + mb * 2048u;
  const unsigned bbase = par_off + (unsigned)(nj * 8) * 8192u;
  while (done != ~0ull) {
    const unsigned va = __hip_atomic_load(fa, __ATOMIC_RELAXED, __HIP_MEMORY_SCOPE_SYSTEM);
    const unsigned vb = __hip_atomic_load(fb, __ATOMIC_RELAXED, __HIP_MEMORY_SCOPE_SYSTEM);
    unsigned long long ready = __ballot(va >= call && vb >= call) & ~done;
    if (ready == 0ull) {
      if (__builtin_amdgcn_readfirstlane(__hip_atomic_load(tmo, __ATOMIC_RELAXED, __HIP_MEMORY_SCOPE_AGENT)) == call ||
          __builtin_amdgcn_s_memrealtime() - t0 > DEADLINE_TICKS) {
        if (lane == 0) __hip_atomic_store(tmo, call, __ATOMIC_RELAXED, __HIP_MEMORY_SCOPE_AGENT);
        failed = true;
        break;
      }
      __builtin_amdgcn_s_sleep(4);
      continue;
    }
    asm volatile("" ::: "memory");
    done |= ready;
    while (ready) {
      const int cnt = __builtin_popcountll(ready);
      if (cnt >= 4) consume<4>(ready, acc, rs, abase, bbase, lane);
      else if (cnt >= 2) consume<2>(ready, acc, rs, abase, bbase, lane);
      else consume<1>(ready, acc, rs, abase, bbase, lane);
    }
  }
  if (__builtin_amdgcn_readfirstlane(__hip_atomic_load(tmo, __ATOMIC_RELAXED, __HIP_MEMORY_SCOPE_AGENT)) == call) failed = true;

  const float scale = failed ? __builtin_nanf("") : P.sx[0] * P.sw[0];
  float* orow = P.out + (size_t)(mi * 64 + mb * 32 + (lane >> 5) * 4) * GN + nj * 128 + (lane & 31) * 4;
#pragma unroll
  for (int r = 0; r < 16; ++r) {
    f32x4 v = {acc[0][r] * scale, acc[1][r] * scale, acc[2][r] * scale, acc[3][r] * scale};
    *reinterpret_cast<f32x4*>(orow + (size_t)((r & 3) + 8 * (r >> 2)) * GN) = v;
  }
}

extern "C" void kernel_launch(int rank, int nranks, ncclComm_t comm,
                              hipStream_t stream,
                              void* const* d_in, const int* in_sizes, int n_in,
                              void* d_out, int out_size,
                              const int* in_dtypes, int out_dtype,
                              void* const* d_out_all,
                              void* const* d_ws_all, size_t ws_bytes) {
  if (nranks != NR || ws_bytes < WS_NEEDED || n_in < 4 || out_size != MS * GN ||
      in_sizes[0] != GM * KS || in_sizes[1] != KS * GN) {
    fprintf(stderr, "kernel_launch: unexpected configuration\n");
    return;
  }
  Params P;
  P.x = static_cast<const unsigned char*>(d_in[0]);
  P.w = static_cast<const unsigned char*>(d_in[1]);
  P.sx = static_cast<const float*>(d_in[2]);
  P.sw = static_cast<const float*>(d_in[3]);
  P.out = static_cast<float*>(d_out);
  for (int p = 0; p < NR; ++p) P.ws[p] = static_cast<unsigned char*>(d_ws_all[p]);
  P.rank = rank;
  gemm_ag_fused<<<256, 256, 0, stream>>>(P);
}
```

```cpp
#include <hip/hip_runtime.h>
#include <rccl/rccl.h>
#include <cstdio>
#include <cstdint>

typedef __attribute__((ext_vector_type(4))) unsigned u32x4;
typedef __attribute__((ext_vector_type(8))) int i32x8;
typedef __attribute__((ext_vector_type(16))) float f32x16;
typedef __attribute__((ext_vector_type(4))) float f32x4;

#define NPW 8
#define UU 2

namespace {

constexpr int NR = 8;
constexpr int GM = 4096, GK = 4096, GN = 4096;
constexpr int MS = GM / NR;
constexpr int KS = GK / NR;

constexpr unsigned CTR_OFF   = 0;
constexpr unsigned TMO_OFF   = 8192;
constexpr unsigned FLAGA_OFF = 16384;
constexpr unsigned FLAGB_OFF = 32768;
constexpr unsigned STAGE_OFF = 1u << 20;
constexpr unsigned A_BYTES   = MS * KS;
constexpr unsigned B_BYTES   = KS * GN;
constexpr unsigned SRC_STRIDE = B_BYTES + A_BYTES;
constexpr unsigned PAR_STRIDE = NR * SRC_STRIDE;
constexpr unsigned WS_NEEDED  = STAGE_OFF + 2 * PAR_STRIDE;
constexpr int NITEM = 288;

constexpr int AUX_SYS = 17;
constexpr unsigned long long DEADLINE_TICKS = 100000000ull;

#define STAMPV(i, v)
#define NOW() 0ull

struct Params {
  const unsigned char* x;
  const unsigned char* w;
  const float* sx;
  const float* sw;
  float* out;
  unsigned char* ws[NR];
  int rank;
};

__device__ __forceinline__ __amdgpu_buffer_rsrc_t make_rsrc(const void* p) {
  return __builtin_amdgcn_make_buffer_rsrc(const_cast<void*>(p), 0, WS_NEEDED, 0x00020000);
}

__device__ __forceinline__ void tr4x4(unsigned d0, unsigned d1, unsigned d2, unsigned d3,
                                      unsigned& t0, unsigned& t1, unsigned& t2, unsigned& t3) {
  const unsigned a01l = __builtin_amdgcn_perm(d1, d0, 0x05010400u);
  const unsigned a01h = __builtin_amdgcn_perm(d1, d0, 0x07030602u);
  const unsigned a23l = __builtin_amdgcn_perm(d3, d2, 0x05010400u);
  const unsigned a23h = __builtin_amdgcn_perm(d3, d2, 0x07030602u);
  t0 = __builtin_amdgcn_perm(a23l, a01l, 0x05040100u);
  t1 = __builtin_amdgcn_perm(a23l, a01l, 0x07060302u);
  t2 = __builtin_amdgcn_perm(a23h, a01h, 0x05040100u);
  t3 = __builtin_amdgcn_perm(a23h, a01h, 0x07060302u);
}

__device__ __forceinline__ i32x8 cat8(u32x4 lo, u32x4 hi) {
  i32x8 r;
  r[0] = lo[0]; r[1] = lo[1]; r[2] = lo[2]; r[3] = lo[3];
  r[4] = hi[0]; r[5] = hi[1]; r[6] = hi[2]; r[7] = hi[3];
  return r;
}

__device__ __forceinline__ void mfma_fp8(f32x16& acc, const i32x8& a, const i32x8& b, int one) {
  asm volatile("s_nop 1\n\tv_mfma_scale_f32_32x32x64_f8f6f4 %0, %1, %2, %0, %3, %3 op_sel_hi:[0,0,0]"
               : "+a"(acc) : "v"(a), "v"(b), "v"(one));
}

template <int U>
__device__ __forceinline__ void consume(unsigned long long& ready, f32x16 (&acc)[4],
                                        __amdgpu_buffer_rsrc_t rs, unsigned abase, unsigned bbase, int lane) {
  u32x4 af[U][2], bf[U][8];
  int one = 0x7f7f7f7f;
  asm volatile("" : "+v"(one));
  const int n = __builtin_popcountll(ready);
#pragma unroll
  for (int t = 0; t < U; ++t) {
    const unsigned u = (unsigned)__builtin_ctzll(ready | (1ull << 63));
    const bool ok = ready != 0ull;
    ready &= ready - 1ull;
    const unsigned so = (u >> 3) * SRC_STRIDE, kc = u & 7u;
    const unsigned ua = __builtin_amdgcn_readfirstlane(ok ? abase + so + kc * (4u * 8192u) : 0x80000000u);
    const unsigned ub = __builtin_amdgcn_readfirstlane(ok ? bbase + so + kc * (8u * 8192u) : 0x80000000u);
#pragma unroll
    for (int q = 0; q < 2; ++q)
      af[t][q] = __builtin_amdgcn_raw_buffer_load_b128(rs, lane * 16 + q * 1024, ua, AUX_SYS);
#pragma unroll
    for (int q = 0; q < 8; ++q)
      bf[t][q] = __builtin_amdgcn_raw_buffer_load_b128(rs, lane * 16 + (q & 3) * 1024, ub + (q >> 2) * 4096u, AUX_SYS);
  }
  __builtin_amdgcn_sched_barrier(0);
#pragma unroll
  for (int t = 0; t < U; ++t) {
    if (t == 0 || n > t) {
      const i32x8 a0 = cat8(af[t][0], af[t][1]);
#pragma unroll
      for (int c = 0; c < 4; ++c) {
        const i32x8 b8 = cat8(bf[t][2 * c], bf[t][2 * c + 1]);
        mfma_fp8(acc[c], a0, b8, one);
      }
    }
  }
}

__device__ __forceinline__ void load_a(u32x4 (&d)[8], int i, const Params& P, unsigned p, int lane) {
  const unsigned char* xs = P.x + (size_t)(p * MS + (i & 3) * 128 + (lane & 31)) * KS + (i >> 2) * 64 + (lane >> 5) * 32;
#pragma unroll
  for (int rb = 0; rb < 4; ++rb)
#pragma unroll
    for (int h = 0; h < 2; ++h)
      d[rb * 2 + h] = *reinterpret_cast<const u32x4*>(xs + (size_t)rb * 32 * KS + h * 16);
}

__device__ __forceinline__ void load_b(unsigned (&d)[32], int j, const Params& P, int lane) {
  const unsigned char* ws = P.w + (size_t)(((j >> 3) & 7) * 64 + (lane >> 5) * 32) * GN + ((j >> 6) * 8 + (j & 7)) * 128 + (lane & 31) * 4;
#pragma unroll
  for (int r = 0; r < 32; ++r) d[r] = *reinterpret_cast<const unsigned*>(ws + (size_t)r * GN);
}

__device__ __forceinline__ void store_a(const u32x4 (&d)[8], int i, __amdgpu_buffer_rsrc_t drs, unsigned sbase, int lane) {
  const unsigned ub = __builtin_amdgcn_readfirstlane(sbase + (unsigned)i * 8192u);
#pragma unroll
  for (int q = 0; q < 8; ++q)
    __builtin_amdgcn_raw_buffer_store_b128(d[q], drs, lane * 16 + (q & 3) * 1024, ub + (q >> 2) * 4096, AUX_SYS);
}

__device__ __forceinline__ void store_b(const unsigned (&d)[32], int j, __amdgpu_buffer_rsrc_t drs, unsigned sbase, int lane) {
  unsigned tt[32];
#pragma unroll
  for (int q = 0; q < 8; ++q)
    tr4x4(d[4 * q], d[4 * q + 1], d[4 * q + 2], d[4 * q + 3], tt[q], tt[8 + q], tt[16 + q], tt[24 + q]);
  const unsigned ub = __builtin_amdgcn_readfirstlane(sbase + A_BYTES + (unsigned)j * 8192u);
#pragma unroll
  for (int q = 0; q < 8; ++q) {
    u32x4 v = {tt[4 * q], tt[4 * q + 1], tt[4 * q + 2], tt[4 * q + 3]};
    __builtin_amdgcn_raw_buffer_store_b128(v, drs, lane * 16 + (q & 3) * 1024, ub + (q >> 2) * 4096, AUX_SYS);
  }
}

__device__ __forceinline__ void publish_b(int j, unsigned char* dws, int rank, unsigned call, int lane) {
  unsigned* f = reinterpret_cast<unsigned*>(dws + FLAGB_OFF) + (((j >> 6) * 8 + (j & 7)) * 64 + rank * 8 + ((j >> 3) & 7));
  if (lane == 0) __hip_atomic_store(f, call, __ATOMIC_RELAXED, __HIP_MEMORY_SCOPE_SYSTEM);
}

}

__global__ void __launch_bounds__(256, 1) gemm_ag_fused(Params P) {
  __shared__ __attribute__((aligned(16))) f32x4 xch[2][4][16][64];
  __shared__ unsigned xflag[4];
  const int lane = threadIdx.x & 63;
  const int wave = __builtin_amdgcn_readfirstlane(threadIdx.x >> 6);
  const int g = blockIdx.x & 7;
  const int wi = blockIdx.x >> 3;
  unsigned char* myws = P.ws[P.rank];
  const unsigned long long tst0 = NOW();

  if (wi < 16) {
    if (wave != 0 || wi >= NPW) return;
    unsigned* ctr = reinterpret_cast<unsigned*>(myws + CTR_OFF) + blockIdx.x * 4;
    const unsigned ctr_v = __hip_atomic_load(ctr, __ATOMIC_RELAXED, __HIP_MEMORY_SCOPE_AGENT);
    const unsigned p = (unsigned)(P.rank + 1 + g) & 7u;
    unsigned char* dws = P.ws[p];
    const __amdgpu_buffer_rsrc_t drs = make_rsrc(dws);
    constexpr int NA = 32 / NPW;
    u32x4 da[NA][8];
    unsigned d0[32], d1[32];
#pragma unroll
    for (int t = 0; t < NA; ++t) load_a(da[t], wi + t * NPW, P, p, lane);
    load_b(d0, wi, P, lane);
    load_b(d1, wi + NPW, P, lane);
    asm volatile("" ::: "memory");
    const unsigned call = __builtin_amdgcn_readfirstlane(ctr_v) + 1u;
    if (lane == 0) __hip_atomic_store(ctr, call, __ATOMIC_RELAXED, __HIP_MEMORY_SCOPE_AGENT);
    const unsigned sbase = STAGE_OFF + (call & 1u) * PAR_STRIDE + (unsigned)P.rank * SRC_STRIDE;
    const unsigned long long tp0 = NOW();
#pragma unroll
    for (int t = 0; t < NA; ++t) store_a(da[t], wi + t * NPW, drs, sbase, lane);
    asm volatile("" ::: "memory");
    store_b(d0, wi, drs, sbase, lane);
    asm volatile("" ::: "memory");
    load_b(d0, wi + 2 * NPW, P, lane);
    asm volatile("s_waitcnt vmcnt(40)" ::: "memory");
    if (lane < NA) {
      const int i = wi + lane * NPW;
      __hip_atomic_store(reinterpret_cast<unsigned*>(dws + FLAGA_OFF) + ((i & 3) * 64 + P.rank * 8 + (i >> 2)), call,
                         __ATOMIC_RELAXED, __HIP_MEMORY_SCOPE_SYSTEM);
    }
    asm volatile("" ::: "memory");
    store_b(d1, wi + NPW, drs, sbase, lane);
    asm volatile("" ::: "memory");
    load_b(d1, wi + 3 * NPW, P, lane);
    int j = wi + 2 * NPW;
    for (; j < 256 - 2 * NPW; j += 2 * NPW) {
      asm volatile("s_waitcnt vmcnt(41)" ::: "memory");
      publish_b(j - 2 * NPW, dws, P.rank, call, lane);
      asm volatile("" ::: "memory");
      store_b(d0, j, drs, sbase, lane);
      asm volatile("" ::: "memory");
      load_b(d0, j + 2 * NPW, P, lane);
      asm volatile("s_waitcnt vmcnt(41)" ::: "memory");
      publish_b(j - NPW, dws, P.rank, call, lane);
      asm volatile("" ::: "memory");
      store_b(d1, j + NPW, drs, sbase, lane);
      asm volatile("" ::: "memory");
      load_b(d1, j + 3 * NPW, P, lane);
    }
    asm volatile("s_waitcnt vmcnt(41)" ::: "memory");
    publish_b(j - 2 * NPW, dws, P.rank, call, lane);
    asm volatile("" ::: "memory");
    store_b(d0, j, drs, sbase, lane);
    asm volatile("s_waitcnt vmcnt(9)" ::: "memory");
    publish_b(j - NPW, dws, P.rank, call, lane);
    asm volatile("" ::: "memory");
    store_b(d1, j + NPW, drs, sbase, lane);
    asm volatile("s_waitcnt vmcnt(9)" ::: "memory");
    publish_b(j, dws, P.rank, call, lane);
    asm volatile("s_waitcnt vmcnt(1)" ::: "memory");
    publish_b(j + NPW, dws, P.rank, call, lane);
    STAMPV(0, tst0); STAMPV(1, tp0); STAMPV(2, NOW());
    return;
  }

  unsigned* ctr = reinterpret_cast<unsigned*>(myws + CTR_OFF) + (blockIdx.x * 4 + wave);
  const unsigned ctr_v = __hip_atomic_load(ctr, __ATOMIC_RELAXED, __HIP_MEMORY_SCOPE_AGENT);
  if (threadIdx.x < 4) xflag[threadIdx.x] = 0u;
  __syncthreads();
  const unsigned call = __builtin_amdgcn_readfirstlane(ctr_v) + 1u;
  if (lane == 0) __hip_atomic_store(ctr, call, __ATOMIC_RELAXED, __HIP_MEMORY_SCOPE_AGENT);
  const unsigned par_off = STAGE_OFF + (call & 1u) * PAR_STRIDE;
  const int gi = wi - 16;
  const int kp = wave;
  const __amdgpu_buffer_rsrc_t rs = make_rsrc(myws);
  const unsigned* fa = reinterpret_cast<const unsigned*>(myws + FLAGA_OFF) + (gi >> 2) * 64 + lane;
  unsigned* tmo = reinterpret_cast<unsigned*>(myws + TMO_OFF);
  const float sc = P.sx[0] * P.sw[0];
  bool failed = false;
  const unsigned long long mine = __ballot((((lane >> 3) + lane) & 3) == kp);
  const unsigned long long t0 = __builtin_amdgcn_s_memrealtime();
  const unsigned abase = par_off + (unsigned)(gi >> 2) * 8192u + (unsigned)(gi & 3) * 2048u;
  unsigned long long tlast = 0, tend = 0; unsigned polls = 0, nlast = 0;

  for (int ph = 0; ph < 4; ++ph) {
    const int nj = 8 * ph + g;
    const unsigned* fb = reinterpret_cast<const unsigned*>(myws + FLAGB_OFF) + nj * 64 + lane;
    const unsigned bbase = par_off + A_BYTES + (unsigned)(ph * 64 + g) * 8192u;
    f32x16 acc[4];
#pragma unroll
    for (int c = 0; c < 4; ++c)
#pragma unroll
      for (int r = 0; r < 16; ++r) acc[c][r] = 0.f;
    unsigned long long done = ~mine;
    unsigned long long ready = 0ull;
    do {
      if (ready == 0ull) {
        do {
          const unsigned va = __hip_atomic_load(fa, __ATOMIC_RELAXED, __HIP_MEMORY_SCOPE_SYSTEM);
          const unsigned vb = __hip_atomic_load(fb, __ATOMIC_RELAXED, __HIP_MEMORY_SCOPE_SYSTEM);
          ready = __ballot(va >= call && vb >= call) & ~done;
          ++polls;
          if (ready == 0ull) {
            __builtin_amdgcn_s_sleep(2);
            if (__builtin_amdgcn_s_memrealtime() - t0 > DEADLINE_TICKS ||
                __builtin_amdgcn_readfirstlane(__hip_atomic_load(tmo, __ATOMIC_RELAXED, __HIP_MEMORY_SCOPE_AGENT)) == call) {
              __hip_atomic_store(tmo, call, __ATOMIC_RELAXED, __HIP_MEMORY_SCOPE_AGENT);
              ready = ~done;
              failed = true;
            }
          }
        } while (ready == 0ull);
        asm volatile("" ::: "memory");
        done |= ready;
        tlast = NOW(); nlast = __builtin_popcountll(ready);
      }
      consume<UU>(ready, acc, rs, abase, bbase, lane);
    } while ((ready | ~done) != 0ull);
    asm volatile("s_nop 15\n\ts_nop 15" : "+a"(acc[0]), "+a"(acc[1]), "+a"(acc[2]), "+a"(acc[3]));
    tend = NOW();

    const float bad = failed ? __builtin_nanf("") : 0.f;
#pragma unroll
    for (int r = 0; r < 16; ++r)
      xch[ph & 1][wave][r][lane] = f32x4{acc[0][r] + bad, acc[1][r], acc[2][r], acc[3][r]};
    __builtin_amdgcn_fence(__ATOMIC_RELEASE, "workgroup");
    if (lane == 0) __hip_atomic_store(&xflag[wave], (unsigned)(ph + 1), __ATOMIC_RELAXED, __HIP_MEMORY_SCOPE_WORKGROUP);
    while (true) {
      const unsigned f = __hip_atomic_load(&xflag[lane & 3], __ATOMIC_RELAXED, __HIP_MEMORY_SCOPE_WORKGROUP);
      if (__all(f >= (unsigned)(ph + 1))) break;
      __builtin_amdgcn_s_sleep(1);
    }
    __builtin_amdgcn_fence(__ATOMIC_ACQUIRE, "workgroup");
    float* orow = P.out + (size_t)(gi * 32 + kp * 8 + (lane >> 5) * 4) * GN + nj * 128 + (lane & 31) * 4;
#pragma unroll
    for (int r = 0; r < 4; ++r) {
      const f32x4 v = (xch[ph & 1][0][4 * kp + r][lane] + xch[ph & 1][1][4 * kp + r][lane]) +
                      (xch[ph & 1][2][4 * kp + r][lane] + xch[ph & 1][3][4 * kp + r][lane]);
      *reinterpret_cast<f32x4*>(orow + (size_t)r * GN) = v * sc;
    }
  }
}

extern "C" void kernel_launch(int rank, int nranks, ncclComm_t comm,
                              hipStream_t stream,
                              void* const* d_in, const int* in_sizes, int n_in,
                              void* d_out, int out_size,
                              const int* in_dtypes, int out_dtype,
                              void* const* d_out_all,
                              void* const* d_ws_all, size_t ws_bytes) {
  if (nranks != NR || ws_bytes < WS_NEEDED || n_in < 4 || out_size != MS * GN ||
      in_sizes[0] != GM * KS || in_sizes[1] != KS * GN) {
    fprintf(stderr, "kernel_launch: unexpected configuration\n");
    return;
  }
  Params P;
  P.x = static_cast<const unsigned char*>(d_in[0]);
  P.w = static_cast<const unsigned char*>(d_in[1]);
  P.sx = static_cast<const float*>(d_in[2]);
  P.sw = static_cast<const float*>(d_in[3]);
  P.out = static_cast<float*>(d_out);
  for (int p = 0; p < NR; ++p) P.ws[p] = static_cast<unsigned char*>(d_ws_all[p]);
  P.rank = rank;
  gemm_ag_fused<<<256, 256, 0, stream>>>(P);
}
```

```cpp
#include <hip/hip_runtime.h>
#include <rccl/rccl.h>
#include <cstdio>
#include <cstdint>
#include <cstdlib>

typedef __attribute__((ext_vector_type(4))) unsigned u32x4;
typedef __attribute__((ext_vector_type(8))) int i32x8;
typedef __attribute__((ext_vector_type(16))) float f32x16;
typedef __attribute__((ext_vector_type(4))) float f32x4;

#define NPW 8
#define UU 2

namespace {

constexpr int NR = 8;
constexpr int GM = 4096, GK = 4096, GN = 4096;
constexpr int MS = GM / NR;
constexpr int KS = GK / NR;

constexpr unsigned CTR_OFF   = 0;
constexpr unsigned TMO_OFF   = 8192;
constexpr unsigned FLAGA_OFF = 16384;
constexpr unsigned FLAGB_OFF = 32768;
constexpr unsigned A_BYTES   = MS * KS;
constexpr unsigned B_BYTES   = KS * GN;
constexpr unsigned BST_OFF   = 2u << 20;
constexpr unsigned AST_OFF   = BST_OFF + 2 * NR * B_BYTES;
constexpr unsigned WS_NEEDED = AST_OFF + 2 * NR * A_BYTES;
constexpr int NITEM = 288;

constexpr int AUX_SYS = 17;
constexpr unsigned long long DEADLINE_TICKS = 100000000ull;

#define STAMPV(i, v)
#define NOW() 0ull

struct Params {
  const unsigned char* x;
  const unsigned char* w;
  const float* sx;
  const float* sw;
  float* out;
  unsigned char* ws[NR];
  int rank;
  unsigned map;
};

__device__ __forceinline__ __amdgpu_buffer_rsrc_t make_rsrc(const void* p) {
  return __builtin_amdgcn_make_buffer_rsrc(const_cast<void*>(p), 0, WS_NEEDED, 0x00020000);
}

__device__ __forceinline__ void tr4x4(unsigned d0, unsigned d1, unsigned d2, unsigned d3,
                                      unsigned& t0, unsigned& t1, unsigned& t2, unsigned& t3) {
  const unsigned a01l = __builtin_amdgcn_perm(d1, d0, 0x05010400u);
  const unsigned a01h = __builtin_amdgcn_perm(d1, d0, 0x07030602u);
  const unsigned a23l = __builtin_amdgcn_perm(d3, d2, 0x05010400u);
  const unsigned a23h = __builtin_amdgcn_perm(d3, d2, 0x07030602u);
  t0 = __builtin_amdgcn_perm(a23l, a01l, 0x05040100u);
  t1 = __builtin_amdgcn_perm(a23l, a01l, 0x07060302u);
  t2 = __builtin_amdgcn_perm(a23h, a01h, 0x05040100u);
  t3 = __builtin_amdgcn_perm(a23h, a01h, 0x07060302u);
}

__device__ __forceinline__ i32x8 cat8(u32x4 lo, u32x4 hi) {
  i32x8 r;
  r[0] = lo[0]; r[1] = lo[1]; r[2] = lo[2]; r[3] = lo[3];
  r[4] = hi[0]; r[5] = hi[1]; r[6] = hi[2]; r[7] = hi[3];
  return r;
}

__device__ __forceinline__ void mfma_fp8(f32x16& acc, const i32x8& a, const i32x8& b, int one) {
  asm volatile("s_nop 1\n\tv_mfma_scale_f32_32x32x64_f8f6f4 %0, %1, %2, %0, %3, %3 op_sel_hi:[0,0,0]"
               : "+a"(acc) : "v"(a), "v"(b), "v"(one));
}

template <int U>
__device__ __forceinline__ void consume(unsigned long long& ready, f32x16 (&acc)[4],
                                        __amdgpu_buffer_rsrc_t rs, unsigned abase, unsigned bbase, int lane) {
  u32x4 af[U][2], bf[U][8];
  int one = 0x7f7f7f7f;
  asm volatile("" : "+v"(one));
  const int n = __builtin_popcountll(ready);
#pragma unroll
  for (int t = 0; t < U; ++t) {
    const unsigned u = (unsigned)__builtin_ctzll(ready | (1ull << 63));
    const bool ok = ready != 0ull;
    ready &= ready - 1ull;
    const unsigned s = u >> 3, kc = u & 7u;
    const unsigned ua = __builtin_amdgcn_readfirstlane(ok ? abase + s * A_BYTES + kc * (4u * 8192u) : 0x80000000u);
    const unsigned ub = __builtin_amdgcn_readfirstlane(ok ? bbase + s * B_BYTES + kc * (8u * 8192u) : 0x80000000u);
#pragma unroll
    for (int q = 0; q < 2; ++q)
      af[t][q] = __builtin_amdgcn_raw_buffer_load_b128(rs, lane * 16 + q * 1024, ua, AUX_SYS);
#pragma unroll
    for (int q = 0; q < 8; ++q)
      bf[t][q] = __builtin_amdgcn_raw_buffer_load_b128(rs, lane * 16 + (q & 3) * 1024, ub + (q >> 2) * 4096u, AUX_SYS);
  }
  __builtin_amdgcn_sched_barrier(0);
#pragma unroll
  for (int t = 0; t < U; ++t) {
    if (t == 0 || n > t) {
      const i32x8 a0 = cat8(af[t][0], af[t][1]);
#pragma unroll
      for (int c = 0; c < 4; ++c) {
        const i32x8 b8 = cat8(bf[t][2 * c], bf[t][2 * c + 1]);
        mfma_fp8(acc[c], a0, b8, one);
      }
    }
  }
}

__device__ __forceinline__ void load_a(u32x4 (&d)[8], int i, const Params& P, unsigned p, int lane) {
  const unsigned char* xs = P.x + (size_t)(p * MS + (i & 3) * 128 + (lane & 31)) * KS + (i >> 2) * 64 + (lane >> 5) * 32;
#pragma unroll
  for (int rb = 0; rb < 4; ++rb)
#pragma unroll
    for (int h = 0; h < 2; ++h)
      d[rb * 2 + h] = *reinterpret_cast<const u32x4*>(xs + (size_t)rb * 32 * KS + h * 16);
}

__device__ __forceinline__ void load_b(unsigned (&d)[32], int j, const Params& P, int lane) {
  const unsigned char* ws = P.w + (size_t)(((j >> 3) & 7) * 64 + (lane >> 5) * 32) * GN + ((j >> 6) * 8 + (j & 7)) * 128 + (lane & 31) * 4;
#pragma unroll
  for (int r = 0; r < 32; ++r) d[r] = *reinterpret_cast<const unsigned*>(ws + (size_t)r * GN);
}

__device__ __forceinline__ void store_a(const u32x4 (&d)[8], int i, __amdgpu_buffer_rsrc_t drs, unsigned sbase, int lane) {
  const unsigned ub = __builtin_amdgcn_readfirstlane(sbase + (unsigned)i * 8192u);
#pragma unroll
  for (int q = 0; q < 8; ++q)
    __builtin_amdgcn_raw_buffer_store_b128(d[q], drs, lane * 16 + (q & 3) * 1024, ub + (q >> 2) * 4096, AUX_SYS);
}

__device__ __forceinline__ void store_b(const unsigned (&d)[32], int j, __amdgpu_buffer_rsrc_t drs, unsigned sbase, int lane) {
  unsigned tt[32];
#pragma unroll
  for (int q = 0; q < 8; ++q)
    tr4x4(d[4 * q], d[4 * q + 1], d[4 * q + 2], d[4 * q + 3], tt[q], tt[8 + q], tt[16 + q], tt[24 + q]);
  const unsigned ub = __builtin_amdgcn_readfirstlane(sbase + (unsigned)j * 8192u);
#pragma unroll
  for (int q = 0; q < 8; ++q) {
    u32x4 v = {tt[4 * q], tt[4 * q + 1], tt[4 * q + 2], tt[4 * q + 3]};
    __builtin_amdgcn_raw_buffer_store_b128(v, drs, lane * 16 + (q & 3) * 1024, ub + (q >> 2) * 4096, AUX_SYS);
  }
}

__device__ __forceinline__ void publish_a(int i, unsigned char* dws, int rank, unsigned call, int lane) {
  unsigned* f = reinterpret_cast<unsigned*>(dws + FLAGA_OFF) + ((i & 3) * 64 + rank * 8 + (i >> 2));
  if (lane == 0) __hip_atomic_store(f, call, __ATOMIC_RELAXED, __HIP_MEMORY_SCOPE_SYSTEM);
}

__device__ __forceinline__ void publish_b(int j, unsigned char* dws, int rank, unsigned call, int lane) {
  unsigned* f = reinterpret_cast<unsigned*>(dws + FLAGB_OFF) + (((j >> 6) * 8 + (j & 7)) * 64 + rank * 8 + ((j >> 3) & 7));
  if (lane == 0) __hip_atomic_store(f, call, __ATOMIC_RELAXED, __HIP_MEMORY_SCOPE_SYSTEM);
}

}

__global__ void __launch_bounds__(256, 1) gemm_ag_fused(Params P) {
  __shared__ __attribute__((aligned(16))) f32x4 xch[2][4][16][64];
  __shared__ unsigned xflag[4];
  const int lane = threadIdx.x & 63;
  const int wave = __builtin_amdgcn_readfirstlane(threadIdx.x >> 6);
  const int g = blockIdx.x & 7;
  const int wi = blockIdx.x >> 3;
  unsigned char* myws = P.ws[P.rank];
  const unsigned long long tst0 = NOW();

  if (wi < 16) {
    if (wave > 0 || wi >= 2 * NPW) return;
    unsigned* ctr = reinterpret_cast<unsigned*>(myws + CTR_OFF) + (blockIdx.x * 4 + wave);
    const unsigned ctr_v = __hip_atomic_load(ctr, __ATOMIC_RELAXED, __HIP_MEMORY_SCOPE_AGENT);
    const unsigned p = (P.map >> (4 * g)) & 7u;
    unsigned char* dws = P.ws[p];
    const __amdgpu_buffer_rsrc_t drs = make_rsrc(dws);
    if (wi >= NPW) {
      static_assert(NPW == 8, "4 x items per wave");
      u32x4 da[4][8];
#pragma unroll
      for (int t = 0; t < 4; ++t) load_a(da[t], wi - NPW + t * NPW, P, p, lane);
      asm volatile("" ::: "memory");
      const unsigned call = __builtin_amdgcn_readfirstlane(ctr_v) + 1u;
      if (lane == 0) __hip_atomic_store(ctr, call, __ATOMIC_RELAXED, __HIP_MEMORY_SCOPE_AGENT);
      const unsigned sbase = AST_OFF + ((call & 1u) * NR + (unsigned)P.rank) * A_BYTES;
      store_a(da[0], wi - NPW, drs, sbase, lane);
      asm volatile("" ::: "memory");
      store_a(da[1], wi, drs, sbase, lane);
      asm volatile("s_waitcnt vmcnt(8)" ::: "memory");
      publish_a(wi - NPW, dws, P.rank, call, lane);
#pragma unroll
      for (int t = 2; t < 4; ++t) {
        asm volatile("" ::: "memory");
        store_a(da[t], wi + (t - 1) * NPW, drs, sbase, lane);
        asm volatile("s_waitcnt vmcnt(9)" ::: "memory");
        publish_a(wi + (t - 2) * NPW, dws, P.rank, call, lane);
      }
      asm volatile("s_waitcnt vmcnt(1)" ::: "memory");
      publish_a(wi + 2 * NPW, dws, P.rank, call, lane);
      return;
    }
    unsigned d0[32], d1[32];
    load_b(d0, wi, P, lane);
    load_b(d1, wi + NPW, P, lane);
    asm volatile("" ::: "memory");
    const unsigned call = __builtin_amdgcn_readfirstlane(ctr_v) + 1u;
    if (lane == 0) __hip_atomic_store(ctr, call, __ATOMIC_RELAXED, __HIP_MEMORY_SCOPE_AGENT);
    const unsigned sbase = BST_OFF + ((call & 1u) * NR + (unsigned)P.rank) * B_BYTES;
    const unsigned long long tp0 = NOW();
    store_b(d0, wi, drs, sbase, lane);
    asm volatile("" ::: "memory");
    load_b(d0, wi + 2 * NPW, P, lane);
    asm volatile("" ::: "memory");
    store_b(d1, wi + NPW, drs, sbase, lane);
    asm volatile("" ::: "memory");
    load_b(d1, wi + 3 * NPW, P, lane);
    asm volatile("s_waitcnt vmcnt(40)" ::: "memory");
    publish_b(wi, dws, P.rank, call, lane);
    int j = wi + 2 * NPW;
    for (; j < 256 - 2 * NPW; j += 2 * NPW) {
      asm volatile("" ::: "memory");
      store_b(d0, j, drs, sbase, lane);
      asm volatile("" ::: "memory");
      load_b(d0, j + 2 * NPW, P, lane);
      asm volatile("s_waitcnt vmcnt(41)" ::: "memory");
      publish_b(j - NPW, dws, P.rank, call, lane);
      asm volatile("" ::: "memory");
      store_b(d1, j + NPW, drs, sbase, lane);
      asm volatile("" ::: "memory");
      load_b(d1, j + 3 * NPW, P, lane);
      asm volatile("s_waitcnt vmcnt(41)" ::: "memory");
      publish_b(j, dws, P.rank, call, lane);
    }
    asm volatile("" ::: "memory");
    store_b(d0, j, drs, sbase, lane);
    asm volatile("s_waitcnt vmcnt(9)" ::: "memory");
    publish_b(j - NPW, dws, P.rank, call, lane);
    asm volatile("" ::: "memory");
    store_b(d1, j + NPW, drs, sbase, lane);
    asm volatile("s_waitcnt vmcnt(9)" ::: "memory");
    publish_b(j, dws, P.rank, call, lane);
    asm volatile("s_waitcnt vmcnt(1)" ::: "memory");
    publish_b(j + NPW, dws, P.rank, call, lane);
    STAMPV(0, tst0); STAMPV(1, tp0); STAMPV(2, NOW());
    return;
  }

  unsigned* ctr = reinterpret_cast<unsigned*>(myws + CTR_OFF) + (blockIdx.x * 4 + wave);
  const unsigned ctr_v = __hip_atomic_load(ctr, __ATOMIC_RELAXED, __HIP_MEMORY_SCOPE_AGENT);
  if (threadIdx.x < 4) xflag[threadIdx.x] = 0u;
  __syncthreads();
  const unsigned call = __builtin_amdgcn_readfirstlane(ctr_v) + 1u;
  if (lane == 0) __hip_atomic_store(ctr, call, __ATOMIC_RELAXED, __HIP_MEMORY_SCOPE_AGENT);
  const int gi = wi - 16;
  const int kp = wave;
  const __amdgpu_buffer_rsrc_t rs = make_rsrc(myws);
  const unsigned* fa = reinterpret_cast<const unsigned*>(myws + FLAGA_OFF) + (gi >> 2) * 64 + lane;
  unsigned* tmo = reinterpret_cast<unsigned*>(myws + TMO_OFF);
  const float sc = P.sx[0] * P.sw[0];
  bool failed = false;
  const unsigned long long mine = __ballot((((lane >> 3) + lane) & 3) == kp);
  const unsigned long long t0 = __builtin_amdgcn_s_memrealtime();
  const unsigned abase = AST_OFF + (call & 1u) * NR * A_BYTES + (unsigned)(gi >> 2) * 8192u + (unsigned)(gi & 3) * 2048u;
  unsigned long long tlast = 0, tend = 0; unsigned polls = 0, nlast = 0;

  for (int ph = 0; ph < 4; ++ph) {
    const int nj = 8 * ph + g;
    const unsigned* fb = reinterpret_cast<const unsigned*>(myws + FLAGB_OFF) + nj * 64 + lane;
    const unsigned bbase = BST_OFF + (call & 1u) * NR * B_BYTES + (unsigned)(ph * 64 + g) * 8192u;
    f32x16 acc[4];
#pragma unroll
    for (int c = 0; c < 4; ++c)
#pragma unroll
      for (int r = 0; r < 16; ++r) acc[c][r] = 0.f;
    unsigned long long done = ~mine;
    unsigned long long ready = 0ull;
    do {
      if (ready == 0ull) {
        do {
          const unsigned va = __hip_atomic_load(fa, __ATOMIC_RELAXED, __HIP_MEMORY_SCOPE_SYSTEM);
          const unsigned vb = __hip_atomic_load(fb, __ATOMIC_RELAXED, __HIP_MEMORY_SCOPE_SYSTEM);
          ready = __ballot(va >= call && vb >= call) & ~done;
          ++polls;
          if (ready == 0ull) {
            __builtin_amdgcn_s_sleep(2);
            if (__builtin_amdgcn_s_memrealtime() - t0 > DEADLINE_TICKS ||
                __builtin_amdgcn_readfirstlane(__hip_atomic_load(tmo, __ATOMIC_RELAXED, __HIP_MEMORY_SCOPE_AGENT)) == call) {
              __hip_atomic_store(tmo, call, __ATOMIC_RELAXED, __HIP_MEMORY_SCOPE_AGENT);
              ready = ~done;
              failed = true;
            }
          }
        } while (ready == 0ull);
        asm volatile("" ::: "memory");
        done |= ready;
        tlast = NOW(); nlast = __builtin_popcountll(ready);
      }
      consume<UU>(ready, acc, rs, abase, bbase, lane);
    } while ((ready | ~done) != 0ull);
    asm volatile("s_nop 15\n\ts_nop 15" : "+a"(acc[0]), "+a"(acc[1]), "+a"(acc[2]), "+a"(acc[3]));
    tend = NOW();

    const float bad = failed ? __builtin_nanf("") : 0.f;
#pragma unroll
    for (int r = 0; r < 16; ++r)
      xch[ph & 1][wave][r][lane] = f32x4{acc[0][r] + bad, acc[1][r], acc[2][r], acc[3][r]};
    __builtin_amdgcn_fence(__ATOMIC_RELEASE, "workgroup");
    if (lane == 0) __hip_atomic_store(&xflag[wave], (unsigned)(ph + 1), __ATOMIC_RELAXED, __HIP_MEMORY_SCOPE_WORKGROUP);
    while (true) {
      const unsigned f = __hip_atomic_load(&xflag[lane & 3], __ATOMIC_RELAXED, __HIP_MEMORY_SCOPE_WORKGROUP);
      if (__all(f >= (unsigned)(ph + 1))) break;
      __builtin_amdgcn_s_sleep(1);
    }
    __builtin_amdgcn_fence(__ATOMIC_ACQUIRE, "workgroup");
    float* orow = P.out + (size_t)(gi * 32 + kp * 8 + (lane >> 5) * 4) * GN + nj * 128 + (lane & 31) * 4;
#pragma unroll
    for (int r = 0; r < 4; ++r) {
      const f32x4 v = (xch[ph & 1][0][4 * kp + r][lane] + xch[ph & 1][1][4 * kp + r][lane]) +
                      (xch[ph & 1][2][4 * kp + r][lane] + xch[ph & 1][3][4 * kp + r][lane]);
      *reinterpret_cast<f32x4*>(orow + (size_t)r * GN) = v * sc;
    }
  }
}

extern "C" void kernel_launch(int rank, int nranks, ncclComm_t comm,
                              hipStream_t stream,
                              void* const* d_in, const int* in_sizes, int n_in,
                              void* d_out, int out_size,
                              const int* in_dtypes, int out_dtype,
                              void* const* d_out_all,
                              void* const* d_ws_all, size_t ws_bytes) {
  if (nranks != NR || ws_bytes < WS_NEEDED || n_in < 4 || out_size != MS * GN ||
      in_sizes[0] != GM * KS || in_sizes[1] != KS * GN) {
    fprintf(stderr, "kernel_launch: unexpected configuration\n");
    return;
  }
  Params P;
  P.x = static_cast<const unsigned char*>(d_in[0]);
  P.w = static_cast<const unsigned char*>(d_in[1]);
  P.sx = static_cast<const float*>(d_in[2]);
  P.sw = static_cast<const float*>(d_in[3]);
  P.out = static_cast<float*>(d_out);
  for (int p = 0; p < NR; ++p) P.ws[p] = static_cast<unsigned char*>(d_ws_all[p]);
  P.rank = rank;
  static const unsigned kMap[NR] = {0x57410362u, 0x62034751u, 0x57416302u, 0x15743602u,
                                    0x60327514u, 0x26307451u, 0x74150623u, 0x20364517u};
  P.map = kMap[rank];
  gemm_ag_fused<<<256, 256, 0, stream>>>(P);
}
```

```cpp
#include <hip/hip_runtime.h>
#include <rccl/rccl.h>
#include <cstdio>
#include <cstdint>
#include <cstdlib>

typedef __attribute__((ext_vector_type(4))) unsigned u32x4;
typedef __attribute__((ext_vector_type(8))) int i32x8;
typedef __attribute__((ext_vector_type(16))) float f32x16;
typedef __attribute__((ext_vector_type(4))) float f32x4;

#define NPW 8
#define UU 2

namespace {

constexpr int NR = 8;
constexpr int GM = 4096, GK = 4096, GN = 4096;
constexpr int MS = GM / NR;
constexpr int KS = GK / NR;

constexpr unsigned CTR_OFF   = 0;
constexpr unsigned TMO_OFF   = 8192;
constexpr unsigned CHK_OFF   = 12288;
constexpr unsigned FLAGA_OFF = 16384;
constexpr unsigned FLAGB_OFF = 32768;
constexpr unsigned A_BYTES   = MS * KS;
constexpr unsigned B_BYTES   = KS * GN;
constexpr unsigned BST_OFF   = 2u << 20;
constexpr unsigned AST_OFF   = BST_OFF + 2 * NR * B_BYTES;
constexpr unsigned WS_NEEDED = AST_OFF + 2 * NR * A_BYTES;
constexpr int NITEM = 288;

constexpr int AUX_SYS = 17;
constexpr unsigned long long DEADLINE_TICKS = 100000000ull;

#define STAMPV(i, v)
#define NOW() 0ull

struct Params {
  const unsigned char* x;
  const unsigned char* w;
  const float* sx;
  const float* sw;
  float* out;
  unsigned char* ws[NR];
  int rank;
  unsigned map;
};

__device__ __forceinline__ __amdgpu_buffer_rsrc_t make_rsrc(const void* p) {
  return __builtin_amdgcn_make_buffer_rsrc(const_cast<void*>(p), 0, WS_NEEDED, 0x00020000);
}

__device__ __forceinline__ void tr4x4(unsigned d0, unsigned d1, unsigned d2, unsigned d3,
                                      unsigned& t0, unsigned& t1, unsigned& t2, unsigned& t3) {
  const unsigned a01l = __builtin_amdgcn_perm(d1, d0, 0x05010400u);
  const unsigned a01h = __builtin_amdgcn_perm(d1, d0, 0x07030602u);
  const unsigned a23l = __builtin_amdgcn_perm(d3, d2, 0x05010400u);
  const unsigned a23h = __builtin_amdgcn_perm(d3, d2, 0x07030602u);
  t0 = __builtin_amdgcn_perm(a23l, a01l, 0x05040100u);
  t1 = __builtin_amdgcn_perm(a23l, a01l, 0x07060302u);
  t2 = __builtin_amdgcn_perm(a23h, a01h, 0x05040100u);
  t3 = __builtin_amdgcn_perm(a23h, a01h, 0x07060302u);
}

__device__ __forceinline__ i32x8 cat8(u32x4 lo, u32x4 hi) {
  i32x8 r;
  r[0] = lo[0]; r[1] = lo[1]; r[2] = lo[2]; r[3] = lo[3];
  r[4] = hi[0]; r[5] = hi[1]; r[6] = hi[2]; r[7] = hi[3];
  return r;
}

__device__ __forceinline__ void mfma_fp8(f32x16& acc, const i32x8& a, const i32x8& b, int one) {
  asm volatile("s_nop 1\n\tv_mfma_scale_f32_32x32x64_f8f6f4 %0, %1, %2, %0, %3, %3 op_sel_hi:[0,0,0]"
               : "+a"(acc) : "v"(a), "v"(b), "v"(one));
}

template <int U>
__device__ __forceinline__ void consume(unsigned long long& ready, f32x16 (&acc)[4],
                                        __amdgpu_buffer_rsrc_t rs, unsigned abase, unsigned bbase, int lane) {
  u32x4 af[U][2], bf[U][8];
  int one = 0x7f7f7f7f;
  asm volatile("" : "+v"(one));
  const int n = __builtin_popcountll(ready);
#pragma unroll
  for (int t = 0; t < U; ++t) {
    const unsigned u = (unsigned)__builtin_ctzll(ready | (1ull << 63));
    const bool ok = ready != 0ull;
    ready &= ready - 1ull;
    const unsigned s = u >> 3, kc = u & 7u;
    const unsigned ua = __builtin_amdgcn_readfirstlane(ok ? abase + s * A_BYTES + kc * (4u * 8192u) : 0x80000000u);
    const unsigned ub = __builtin_amdgcn_readfirstlane(ok ? bbase + s * B_BYTES + kc * (8u * 8192u) : 0x80000000u);
#pragma unroll
    for (int q = 0; q < 2; ++q)
      af[t][q] = __builtin_amdgcn_raw_buffer_load_b128(rs, lane * 16 + q * 1024, ua, AUX_SYS);
#pragma unroll
    for (int q = 0; q < 8; ++q)
      bf[t][q] = __builtin_amdgcn_raw_buffer_load_b128(rs, lane * 16 + (q & 3) * 1024, ub + (q >> 2) * 4096u, AUX_SYS);
  }
  __builtin_amdgcn_sched_barrier(0);
#pragma unroll
  for (int t = 0; t < U; ++t) {
    if (t == 0 || n > t) {
      const i32x8 a0 = cat8(af[t][0], af[t][1]);
#pragma unroll
      for (int c = 0; c < 4; ++c) {
        const i32x8 b8 = cat8(bf[t][2 * c], bf[t][2 * c + 1]);
        mfma_fp8(acc[c], a0, b8, one);
      }
    }
  }
}

__device__ __forceinline__ void load_a(u32x4 (&d)[8], int i, const Params& P, unsigned p, int lane) {
  const unsigned char* xs = P.x + (size_t)(p * MS + (i & 3) * 128 + (lane & 31)) * KS + (i >> 2) * 64 + (lane >> 5) * 32;
#pragma unroll
  for (int rb = 0; rb < 4; ++rb)
#pragma unroll
    for (int h = 0; h < 2; ++h)
      d[rb * 2 + h] = *reinterpret_cast<const u32x4*>(xs + (size_t)rb * 32 * KS + h * 16);
}

__device__ __forceinline__ void load_b(unsigned (&d)[32], int j, const Params& P, int lane) {
  const unsigned char* ws = P.w + (size_t)(((j >> 3) & 7) * 64 + (lane >> 5) * 32) * GN + ((j >> 6) * 8 + (j & 7)) * 128 + (lane & 31) * 4;
#pragma unroll
  for (int r = 0; r < 32; ++r) d[r] = *reinterpret_cast<const unsigned*>(ws + (size_t)r * GN);
}

__device__ __forceinline__ void store_a(const u32x4 (&d)[8], int i, __amdgpu_buffer_rsrc_t drs, unsigned sbase, int lane) {
  const unsigned ub = __builtin_amdgcn_readfirstlane(sbase + (unsigned)i * 8192u);
#pragma unroll
  for (int q = 0; q < 8; ++q)
    __builtin_amdgcn_raw_buffer_store_b128(d[q], drs, lane * 16 + (q & 3) * 1024, ub + (q >> 2) * 4096, AUX_SYS);
}

__device__ __forceinline__ void store_b(const unsigned (&d)[32], int j, __amdgpu_buffer_rsrc_t drs, unsigned sbase, int lane) {
  unsigned tt[32];
#pragma unroll
  for (int q = 0; q < 8; ++q)
    tr4x4(d[4 * q], d[4 * q + 1], d[4 * q + 2], d[4 * q + 3], tt[q], tt[8 + q], tt[16 + q], tt[24 + q]);
  const unsigned ub = __builtin_amdgcn_readfirstlane(sbase + (unsigned)j * 8192u);
#pragma unroll
  for (int q = 0; q < 8; ++q) {
    u32x4 v = {tt[4 * q], tt[4 * q + 1], tt[4 * q + 2], tt[4 * q + 3]};
    __builtin_amdgcn_raw_buffer_store_b128(v, drs, lane * 16 + (q & 3) * 1024, ub + (q >> 2) * 4096, AUX_SYS);
  }
}

__device__ __forceinline__ void publish_a(int i, __amdgpu_buffer_rsrc_t drs, int rank, unsigned call, int lane) {
  const unsigned off = FLAGA_OFF + 4u * (unsigned)((i & 3) * 64 + rank * 8 + (i >> 2));
  if (lane == 0) __builtin_amdgcn_raw_buffer_store_b32(call, drs, 0, __builtin_amdgcn_readfirstlane(off), AUX_SYS);
}

__device__ __forceinline__ void publish_b(int j, __amdgpu_buffer_rsrc_t drs, int rank, unsigned call, int lane) {
  const unsigned off = FLAGB_OFF + 4u * (unsigned)(((j >> 6) * 8 + (j & 7)) * 64 + rank * 8 + ((j >> 3) & 7));
  if (lane == 0) __builtin_amdgcn_raw_buffer_store_b32(call, drs, 0, __builtin_amdgcn_readfirstlane(off), AUX_SYS);
}

__device__ __forceinline__ void push_role(const Params& P, unsigned gg, int wi, unsigned ctr_v, int lane) {
  const unsigned p = (P.map >> (4u * gg)) & 7u;
  const unsigned long long dv = reinterpret_cast<unsigned long long>(P.ws[p]);
  const __amdgpu_buffer_rsrc_t drs = make_rsrc(reinterpret_cast<void*>(
      ((unsigned long long)(unsigned)__builtin_amdgcn_readfirstlane((unsigned)(dv >> 32)) << 32) |
      (unsigned long long)(unsigned)__builtin_amdgcn_readfirstlane((unsigned)dv)));
  if (wi >= NPW) {
    static_assert(NPW == 8, "4 x items per wave");
    u32x4 da[4][8];
#pragma unroll
    for (int t = 0; t < 4; ++t) load_a(da[t], wi - NPW + t * NPW, P, p, lane);
    asm volatile("" ::: "memory");
    const unsigned call = __builtin_amdgcn_readfirstlane(ctr_v) + 1u;
    const unsigned sbase = AST_OFF + ((call & 1u) * NR + (unsigned)P.rank) * A_BYTES;
    store_a(da[0], wi - NPW, drs, sbase, lane);
    asm volatile("" ::: "memory");
    store_a(da[1], wi, drs, sbase, lane);
    asm volatile("s_waitcnt vmcnt(8)" ::: "memory");
    publish_a(wi - NPW, drs, P.rank, call, lane);
#pragma unroll
    for (int t = 2; t < 4; ++t) {
      asm volatile("" ::: "memory");
      store_a(da[t], wi + (t - 1) * NPW, drs, sbase, lane);
      asm volatile("s_waitcnt vmcnt(9)" ::: "memory");
      publish_a(wi + (t - 2) * NPW, drs, P.rank, call, lane);
    }
    asm volatile("s_waitcnt vmcnt(1)" ::: "memory");
    publish_a(wi + 2 * NPW, drs, P.rank, call, lane);
  } else {
    unsigned d0[32], d1[32];
    load_b(d0, wi, P, lane);
    load_b(d1, wi + NPW, P, lane);
    asm volatile("" ::: "memory");
    const unsigned call = __builtin_amdgcn_readfirstlane(ctr_v) + 1u;
    const unsigned sbase = BST_OFF + ((call & 1u) * NR + (unsigned)P.rank) * B_BYTES;
    store_b(d0, wi, drs, sbase, lane);
    asm volatile("" ::: "memory");
    load_b(d0, wi + 2 * NPW, P, lane);
    asm volatile("" ::: "memory");
    store_b(d1, wi + NPW, drs, sbase, lane);
    asm volatile("" ::: "memory");
    load_b(d1, wi + 3 * NPW, P, lane);
    asm volatile("s_waitcnt vmcnt(40)" ::: "memory");
    publish_b(wi, drs, P.rank, call, lane);
    int j = wi + 2 * NPW;
    for (; j < 256 - 2 * NPW; j += 2 * NPW) {
      asm volatile("" ::: "memory");
      store_b(d0, j, drs, sbase, lane);
      asm volatile("" ::: "memory");
      load_b(d0, j + 2 * NPW, P, lane);
      asm volatile("s_waitcnt vmcnt(41)" ::: "memory");
      publish_b(j - NPW, drs, P.rank, call, lane);
      asm volatile("" ::: "memory");
      store_b(d1, j + NPW, drs, sbase, lane);
      asm volatile("" ::: "memory");
      load_b(d1, j + 3 * NPW, P, lane);
      asm volatile("s_waitcnt vmcnt(41)" ::: "memory");
      publish_b(j, drs, P.rank, call, lane);
    }
    asm volatile("" ::: "memory");
    store_b(d0, j, drs, sbase, lane);
    asm volatile("s_waitcnt vmcnt(9)" ::: "memory");
    publish_b(j - NPW, drs, P.rank, call, lane);
    asm volatile("" ::: "memory");
    store_b(d1, j + NPW, drs, sbase, lane);
    asm volatile("s_waitcnt vmcnt(9)" ::: "memory");
    publish_b(j, drs, P.rank, call, lane);
    asm volatile("s_waitcnt vmcnt(1)" ::: "memory");
    publish_b(j + NPW, drs, P.rank, call, lane);
  }
  asm volatile("" ::: "memory");
}

}

__global__ void __launch_bounds__(256, 1) gemm_ag_fused(Params P) {
  __shared__ __attribute__((aligned(16))) f32x4 xch[2][4][16][64];
  __shared__ unsigned xflag[4];
  const int lane = threadIdx.x & 63;
  const int wave = __builtin_amdgcn_readfirstlane(threadIdx.x >> 6);
  const int g = blockIdx.x & 7;
  const int wi = blockIdx.x >> 3;
  unsigned char* myws = P.ws[P.rank];
  const unsigned long long tst0 = NOW();

  if (wi < 16) {
    if (wave > 1) return;
    unsigned* ctr = reinterpret_cast<unsigned*>(myws + CTR_OFF) + (blockIdx.x * 4 + wave);
    const unsigned ctr_v = __hip_atomic_load(ctr, __ATOMIC_RELAXED, __HIP_MEMORY_SCOPE_AGENT);
    unsigned xcc;
    asm volatile("s_getreg_b32 %0, hwreg(HW_REG_XCC_ID)" : "=s"(xcc));
    xcc &= 7u;
    if (wave == 0) {
      push_role(P, xcc, wi, ctr_v, lane);
      if (lane == 0)
        __hip_atomic_store(ctr, __builtin_amdgcn_readfirstlane(ctr_v) + 1u, __ATOMIC_RELAXED, __HIP_MEMORY_SCOPE_AGENT);
      if (wi < NPW) { STAMPV(0, tst0); STAMPV(1, tst0); STAMPV(2, NOW()); }
      return;
    }
    const unsigned call = __builtin_amdgcn_readfirstlane(ctr_v) + 1u;
    if (lane == 0) __hip_atomic_store(ctr, call, __ATOMIC_RELAXED, __HIP_MEMORY_SCOPE_AGENT);
    unsigned long long* chk = reinterpret_cast<unsigned long long*>(myws + CHK_OFF) + wi * 32;
    unsigned long long v = 0ull;
    if (lane == 0) v = __hip_atomic_fetch_add(chk + (call & 1u) * 16, (1ull << 32) + (1ull << (4u * xcc)),
                                              __ATOMIC_RELAXED, __HIP_MEMORY_SCOPE_AGENT);
    const unsigned lo = __builtin_amdgcn_readfirstlane((unsigned)v) + (1u << (4u * xcc));
    const unsigned cnt = __builtin_amdgcn_readfirstlane((unsigned)(v >> 32));
    if (cnt == 7u) {
      if (lane == 0) __hip_atomic_store(chk + ((call + 1u) & 1u) * 16, 0ull, __ATOMIC_RELAXED, __HIP_MEMORY_SCOPE_AGENT);
#pragma nounroll
      for (unsigned q = 0; q < 8u; ++q)
        if (((lo >> (4u * q)) & 15u) == 0u) push_role(P, q, wi, ctr_v, lane);
    }
    return;
  }

  unsigned* ctr = reinterpret_cast<unsigned*>(myws + CTR_OFF) + (blockIdx.x * 4 + wave);
  const unsigned ctr_v = __hip_atomic_load(ctr, __ATOMIC_RELAXED, __HIP_MEMORY_SCOPE_AGENT);
  if (threadIdx.x < 4) xflag[threadIdx.x] = 0u;
  __syncthreads();
  const unsigned call = __builtin_amdgcn_readfirstlane(ctr_v) + 1u;
  if (lane == 0) __hip_atomic_store(ctr, call, __ATOMIC_RELAXED, __HIP_MEMORY_SCOPE_AGENT);
  const int gi = wi - 16;
  const int kp = wave;
  const __amdgpu_buffer_rsrc_t rs = make_rsrc(myws);
  const unsigned* fa = reinterpret_cast<const unsigned*>(myws + FLAGA_OFF) + (gi >> 2) * 64 + lane;
  unsigned* tmo = reinterpret_cast<unsigned*>(myws + TMO_OFF);
  const float sc = P.sx[0] * P.sw[0];
  bool failed = false;
  const unsigned long long mine = __ballot((((lane >> 3) + lane) & 3) == kp);
  const unsigned long long t0 = __builtin_amdgcn_s_memrealtime();
  const unsigned abase = AST_OFF + (call & 1u) * NR * A_BYTES + (unsigned)(gi >> 2) * 8192u + (unsigned)(gi & 3) * 2048u;
  unsigned long long tlast = 0, tend = 0; unsigned polls = 0, nlast = 0;

  for (int ph = 0; ph < 4; ++ph) {
    const int nj = 8 * ph + g;
    const unsigned* fb = reinterpret_cast<const unsigned*>(myws + FLAGB_OFF) + nj * 64 + lane;
    const unsigned bbase = BST_OFF + (call & 1u) * NR * B_BYTES + (unsigned)(ph * 64 + g) * 8192u;
    f32x16 acc[4];
#pragma unroll
    for (int c = 0; c < 4; ++c)
#pragma unroll
      for (int r = 0; r < 16; ++r) acc[c][r] = 0.f;
    unsigned long long done = ~mine;
    unsigned long long ready = 0ull;
    do {
      if (ready == 0ull) {
        do {
          const unsigned va = __hip_atomic_load(fa, __ATOMIC_RELAXED, __HIP_MEMORY_SCOPE_SYSTEM);
          const unsigned vb = __hip_atomic_load(fb, __ATOMIC_RELAXED, __HIP_MEMORY_SCOPE_SYSTEM);
          ready = __ballot(va >= call && vb >= call) & ~done;
          ++polls;
          if (ready == 0ull) {
            __builtin_amdgcn_s_sleep(2);
            if (__builtin_amdgcn_s_memrealtime() - t0 > DEADLINE_TICKS ||
                __builtin_amdgcn_readfirstlane(__hip_atomic_load(tmo, __ATOMIC_RELAXED, __HIP_MEMORY_SCOPE_AGENT)) == call) {
              __hip_atomic_store(tmo, call, __ATOMIC_RELAXED, __HIP_MEMORY_SCOPE_AGENT);
              ready = ~done;
              failed = true;
            }
          }
        } while (ready == 0ull);
        asm volatile("" ::: "memory");
        done |= ready;
        tlast = NOW(); nlast = __builtin_popcountll(ready);
      }
      consume<UU>(ready, acc, rs, abase, bbase, lane);
    } while ((ready | ~done) != 0ull);
    asm volatile("s_nop 15\n\ts_nop 15" : "+a"(acc[0]), "+a"(acc[1]), "+a"(acc[2]), "+a"(acc[3]));
    tend = NOW();

    const float bad = failed ? __builtin_nanf("") : 0.f;
#pragma unroll
    for (int r = 0; r < 16; ++r)
      xch[ph & 1][wave][r][lane] = f32x4{acc[0][r] + bad, acc[1][r], acc[2][r], acc[3][r]};
    __builtin_amdgcn_fence(__ATOMIC_RELEASE, "workgroup");
    if (lane == 0) __hip_atomic_store(&xflag[wave], (unsigned)(ph + 1), __ATOMIC_RELAXED, __HIP_MEMORY_SCOPE_WORKGROUP);
    while (true) {
      const unsigned f = __hip_atomic_load(&xflag[lane & 3], __ATOMIC_RELAXED, __HIP_MEMORY_SCOPE_WORKGROUP);
      if (__all(f >= (unsigned)(ph + 1))) break;
      __builtin_amdgcn_s_sleep(1);
    }
    __builtin_amdgcn_fence(__ATOMIC_ACQUIRE, "workgroup");
    float* orow = P.out + (size_t)(gi * 32 + kp * 8 + (lane >> 5) * 4) * GN + nj * 128 + (lane & 31) * 4;
#pragma unroll
    for (int r = 0; r < 4; ++r) {
      const f32x4 v = (xch[ph & 1][0][4 * kp + r][lane] + xch[ph & 1][1][4 * kp + r][lane]) +
                      (xch[ph & 1][2][4 * kp + r][lane] + xch[ph & 1][3][4 * kp + r][lane]);
      *reinterpret_cast<f32x4*>(orow + (size_t)r * GN) = v * sc;
    }
  }
}

extern "C" void kernel_launch(int rank, int nranks, ncclComm_t comm,
                              hipStream_t stream,
                              void* const* d_in, const int* in_sizes, int n_in,
                              void* d_out, int out_size,
                              const int* in_dtypes, int out_dtype,
                              void* const* d_out_all,
                              void* const* d_ws_all, size_t ws_bytes) {
  if (nranks != NR || ws_bytes < WS_NEEDED || n_in < 4 || out_size != MS * GN ||
      in_sizes[0] != GM * KS || in_sizes[1] != KS * GN) {
    fprintf(stderr, "kernel_launch: unexpected configuration\n");
    return;
  }
  Params P;
  P.x = static_cast<const unsigned char*>(d_in[0]);
  P.w = static_cast<const unsigned char*>(d_in[1]);
  P.sx = static_cast<const float*>(d_in[2]);
  P.sw = static_cast<const float*>(d_in[3]);
  P.out = static_cast<float*>(d_out);
  for (int p = 0; p < NR; ++p) P.ws[p] = static_cast<unsigned char*>(d_ws_all[p]);
  P.rank = rank;
  static const unsigned kMap[NR] = {0x57410362u, 0x62034751u, 0x57416302u, 0x15743602u,
                                    0x60327514u, 0x26307451u, 0x74150623u, 0x20364517u};
  P.map = kMap[rank];
  gemm_ag_fused<<<256, 256, 0, stream>>>(P);
}
```

```cpp
#include <hip/hip_runtime.h>
#include <rccl/rccl.h>
#include <cstdio>
#include <cstdint>

typedef __attribute__((ext_vector_type(4))) unsigned u32x4;
typedef __attribute__((ext_vector_type(8))) int i32x8;
typedef __attribute__((ext_vector_type(16))) float f32x16;
typedef __attribute__((ext_vector_type(4))) float f32x4;


namespace {

constexpr int NR = 8;
constexpr int GM = 4096, GK = 4096, GN = 4096;
constexpr int MS = GM / NR;
constexpr int KS = GK / NR;

constexpr unsigned CTR_OFF   = 0;
constexpr unsigned TMO_OFF   = 8192;
constexpr unsigned CHK_OFF   = 12288;
constexpr unsigned FLAGA_OFF = 16384;
constexpr unsigned FLAGB_OFF = 32768;
constexpr unsigned A_BYTES   = MS * KS;
constexpr unsigned B_BYTES   = KS * GN;
constexpr unsigned BST_OFF   = 2u << 20;
constexpr unsigned AST_OFF   = BST_OFF + 2 * NR * B_BYTES;
constexpr unsigned WS_NEEDED = AST_OFF + 2 * NR * A_BYTES;
constexpr int NPW = 8;
constexpr int UU = 2;

constexpr int AUX_SYS = 17;
constexpr unsigned long long DEADLINE_TICKS = 100000000ull;

struct Params {
  const unsigned char* x;
  const unsigned char* w;
  const float* sx;
  const float* sw;
  float* out;
  unsigned char* ws[NR];
  int rank;
  unsigned map;
};

__device__ __forceinline__ __amdgpu_buffer_rsrc_t make_rsrc(const void* p) {
  return __builtin_amdgcn_make_buffer_rsrc(const_cast<void*>(p), 0, WS_NEEDED, 0x00020000);
}

__device__ __forceinline__ void tr4x4(unsigned d0, unsigned d1, unsigned d2, unsigned d3,
                                      unsigned& t0, unsigned& t1, unsigned& t2, unsigned& t3) {
  const unsigned a01l = __builtin_amdgcn_perm(d1, d0, 0x05010400u);
  const unsigned a01h = __builtin_amdgcn_perm(d1, d0, 0x07030602u);
  const unsigned a23l = __builtin_amdgcn_perm(d3, d2, 0x05010400u);
  const unsigned a23h = __builtin_amdgcn_perm(d3, d2, 0x07030602u);
  t0 = __builtin_amdgcn_perm(a23l, a01l, 0x05040100u);
  t1 = __builtin_amdgcn_perm(a23l, a01l, 0x07060302u);
  t2 = __builtin_amdgcn_perm(a23h, a01h, 0x05040100u);
  t3 = __builtin_amdgcn_perm(a23h, a01h, 0x07060302u);
}

__device__ __forceinline__ i32x8 cat8(u32x4 lo, u32x4 hi) {
  i32x8 r;
  r[0] = lo[0]; r[1] = lo[1]; r[2] = lo[2]; r[3] = lo[3];
  r[4] = hi[0]; r[5] = hi[1]; r[6] = hi[2]; r[7] = hi[3];
  return r;
}

__device__ __forceinline__ void mfma_fp8(f32x16& acc, const i32x8& a, const i32x8& b, int one) {
  asm volatile("s_nop 1\n\tv_mfma_scale_f32_32x32x64_f8f6f4 %0, %1, %2, %0, %3, %3 op_sel_hi:[0,0,0]"
               : "+a"(acc) : "v"(a), "v"(b), "v"(one));
}

template <int U>
__device__ __forceinline__ void consume(unsigned long long& ready, f32x16 (&acc)[4],
                                        __amdgpu_buffer_rsrc_t rs, unsigned abase, unsigned bbase, int lane) {
  u32x4 af[U][2], bf[U][8];
  int one = 0x7f7f7f7f;
  asm volatile("" : "+v"(one));
  const int n = __builtin_popcountll(ready);
#pragma unroll
  for (int t = 0; t < U; ++t) {
    const unsigned u = (unsigned)__builtin_ctzll(ready | (1ull << 63));
    const bool ok = ready != 0ull;
    ready &= ready - 1ull;
    const unsigned s = u >> 3, kc = u & 7u;
    const unsigned ua = __builtin_amdgcn_readfirstlane(ok ? abase + s * A_BYTES + kc * (4u * 8192u) : 0x80000000u);
    const unsigned ub = __builtin_amdgcn_readfirstlane(ok ? bbase + s * B_BYTES + kc * (8u * 8192u) : 0x80000000u);
#pragma unroll
    for (int q = 0; q < 2; ++q)
      af[t][q] = __builtin_amdgcn_raw_buffer_load_b128(rs, lane * 16 + q * 1024, ua, AUX_SYS);
#pragma unroll
    for (int q = 0; q < 8; ++q)
      bf[t][q] = __builtin_amdgcn_raw_buffer_load_b128(rs, lane * 16 + (q & 3) * 1024, ub + (q >> 2) * 4096u, AUX_SYS);
  }
  __builtin_amdgcn_sched_barrier(0);
#pragma unroll
  for (int t = 0; t < U; ++t) {
    if (t == 0 || n > t) {
      const i32x8 a0 = cat8(af[t][0], af[t][1]);
#pragma unroll
      for (int c = 0; c < 4; ++c) {
        const i32x8 b8 = cat8(bf[t][2 * c], bf[t][2 * c + 1]);
        mfma_fp8(acc[c], a0, b8, one);
      }
    }
  }
}

__device__ __forceinline__ void load_a(u32x4 (&d)[8], int i, const Params& P, unsigned p, int lane) {
  const unsigned char* xs = P.x + (size_t)(p * MS + (i & 3) * 128 + (lane & 31)) * KS + (i >> 2) * 64 + (lane >> 5) * 32;
#pragma unroll
  for (int rb = 0; rb < 4; ++rb)
#pragma unroll
    for (int h = 0; h < 2; ++h)
      d[rb * 2 + h] = *reinterpret_cast<const u32x4*>(xs + (size_t)rb * 32 * KS + h * 16);
}

__device__ __forceinline__ void load_b(unsigned (&d)[32], int j, const Params& P, int lane) {
  const unsigned char* ws = P.w + (size_t)(((j >> 3) & 7) * 64 + (lane >> 5) * 32) * GN + ((j >> 6) * 8 + (j & 7)) * 128 + (lane & 31) * 4;
#pragma unroll
  for (int r = 0; r < 32; ++r) d[r] = *reinterpret_cast<const unsigned*>(ws + (size_t)r * GN);
}

__device__ __forceinline__ void store_a(const u32x4 (&d)[8], int i, __amdgpu_buffer_rsrc_t drs, unsigned sbase, int lane) {
  const unsigned ub = __builtin_amdgcn_readfirstlane(sbase + (unsigned)i * 8192u);
#pragma unroll
  for (int q = 0; q < 8; ++q)
    __builtin_amdgcn_raw_buffer_store_b128(d[q], drs, lane * 16 + (q & 3) * 1024, ub + (q >> 2) * 4096, AUX_SYS);
}

__device__ __forceinline__ void store_b(const unsigned (&d)[32], int j, __amdgpu_buffer_rsrc_t drs, unsigned sbase, int lane) {
  unsigned tt[32];
#pragma unroll
  for (int q = 0; q < 8; ++q)
    tr4x4(d[4 * q], d[4 * q + 1], d[4 * q + 2], d[4 * q + 3], tt[q], tt[8 + q], tt[16 + q], tt[24 + q]);
  const unsigned ub = __builtin_amdgcn_readfirstlane(sbase + (unsigned)j * 8192u);
#pragma unroll
  for (int q = 0; q < 8; ++q) {
    u32x4 v = {tt[4 * q], tt[4 * q + 1], tt[4 * q + 2], tt[4 * q + 3]};
    __builtin_amdgcn_raw_buffer_store_b128(v, drs, lane * 16 + (q & 3) * 1024, ub + (q >> 2) * 4096, AUX_SYS);
  }
}

__device__ __forceinline__ void publish_a(int i, __amdgpu_buffer_rsrc_t drs, int rank, unsigned call, int lane) {
  const unsigned off = FLAGA_OFF + 4u * (unsigned)((i & 3) * 64 + rank * 8 + (i >> 2));
  if (lane == 0) __builtin_amdgcn_raw_buffer_store_b32(call, drs, 0, __builtin_amdgcn_readfirstlane(off), AUX_SYS);
}

__device__ __forceinline__ void publish_b(int j, __amdgpu_buffer_rsrc_t drs, int rank, unsigned call, int lane) {
  const unsigned off = FLAGB_OFF + 4u * (unsigned)(((j >> 6) * 8 + (j & 7)) * 64 + rank * 8 + ((j >> 3) & 7));
  if (lane == 0) __builtin_amdgcn_raw_buffer_store_b32(call, drs, 0, __builtin_amdgcn_readfirstlane(off), AUX_SYS);
}

__device__ __forceinline__ void push_role(const Params& P, unsigned gg, int wi, unsigned ctr_v, int lane) {
  const unsigned p = (P.map >> (4u * gg)) & 7u;
  const unsigned long long dv = reinterpret_cast<unsigned long long>(P.ws[p]);
  const __amdgpu_buffer_rsrc_t drs = make_rsrc(reinterpret_cast<void*>(
      ((unsigned long long)(unsigned)__builtin_amdgcn_readfirstlane((unsigned)(dv >> 32)) << 32) |
      (unsigned long long)(unsigned)__builtin_amdgcn_readfirstlane((unsigned)dv)));
  if (wi >= NPW) {
    static_assert(NPW == 8, "4 x items per wave");
    u32x4 da[4][8];
#pragma unroll
    for (int t = 0; t < 4; ++t) load_a(da[t], wi - NPW + t * NPW, P, p, lane);
    asm volatile("" ::: "memory");
    const unsigned call = __builtin_amdgcn_readfirstlane(ctr_v) + 1u;
    const unsigned sbase = AST_OFF + ((call & 1u) * NR + (unsigned)P.rank) * A_BYTES;
    store_a(da[0], wi - NPW, drs, sbase, lane);
    asm volatile("" ::: "memory");
    store_a(da[1], wi, drs, sbase, lane);
    asm volatile("s_waitcnt vmcnt(8)" ::: "memory");
    publish_a(wi - NPW, drs, P.rank, call, lane);
#pragma unroll
    for (int t = 2; t < 4; ++t) {
      asm volatile("" ::: "memory");
      store_a(da[t], wi + (t - 1) * NPW, drs, sbase, lane);
      asm volatile("s_waitcnt vmcnt(9)" ::: "memory");
      publish_a(wi + (t - 2) * NPW, drs, P.rank, call, lane);
    }
    asm volatile("s_waitcnt vmcnt(1)" ::: "memory");
    publish_a(wi + 2 * NPW, drs, P.rank, call, lane);
  } else {
    unsigned d0[32], d1[32];
    load_b(d0, wi, P, lane);
    load_b(d1, wi + NPW, P, lane);
    asm volatile("" ::: "memory");
    const unsigned call = __builtin_amdgcn_readfirstlane(ctr_v) + 1u;
    const unsigned sbase = BST_OFF + ((call & 1u) * NR + (unsigned)P.rank) * B_BYTES;
    store_b(d0, wi, drs, sbase, lane);
    asm volatile("" ::: "memory");
    load_b(d0, wi + 2 * NPW, P, lane);
    asm volatile("" ::: "memory");
    store_b(d1, wi + NPW, drs, sbase, lane);
    asm volatile("" ::: "memory");
    load_b(d1, wi + 3 * NPW, P, lane);
    asm volatile("s_waitcnt vmcnt(40)" ::: "memory");
    publish_b(wi, drs, P.rank, call, lane);
    int j = wi + 2 * NPW;
    for (; j < 256 - 2 * NPW; j += 2 * NPW) {
      asm volatile("" ::: "memory");
      store_b(d0, j, drs, sbase, lane);
      asm volatile("" ::: "memory");
      load_b(d0, j + 2 * NPW, P, lane);
      asm volatile("s_waitcnt vmcnt(41)" ::: "memory");
      publish_b(j - NPW, drs, P.rank, call, lane);
      asm volatile("" ::: "memory");
      store_b(d1, j + NPW, drs, sbase, lane);
      asm volatile("" ::: "memory");
      load_b(d1, j + 3 * NPW, P, lane);
      asm volatile("s_waitcnt vmcnt(41)" ::: "memory");
      publish_b(j, drs, P.rank, call, lane);
    }
    asm volatile("" ::: "memory");
    store_b(d0, j, drs, sbase, lane);
    asm volatile("s_waitcnt vmcnt(9)" ::: "memory");
    publish_b(j - NPW, drs, P.rank, call, lane);
    asm volatile("" ::: "memory");
    store_b(d1, j + NPW, drs, sbase, lane);
    asm volatile("s_waitcnt vmcnt(9)" ::: "memory");
    publish_b(j, drs, P.rank, call, lane);
    asm volatile("s_waitcnt vmcnt(1)" ::: "memory");
    publish_b(j + NPW, drs, P.rank, call, lane);
  }
  asm volatile("" ::: "memory");
}

}

__global__ void __launch_bounds__(256, 1) gemm_ag_fused(Params P) {
  __shared__ __attribute__((aligned(16))) f32x4 xch[2][4][16][64];
  __shared__ unsigned xflag[4];
  const int lane = threadIdx.x & 63;
  const int wave = __builtin_amdgcn_readfirstlane(threadIdx.x >> 6);
  const int g = blockIdx.x & 7;
  const int wi = blockIdx.x >> 3;
  unsigned char* myws = P.ws[P.rank];

  if (wi < 16) {
    if (wave > 1) return;
    unsigned* ctr = reinterpret_cast<unsigned*>(myws + CTR_OFF) + (blockIdx.x * 4 + wave);
    const unsigned ctr_v = __hip_atomic_load(ctr, __ATOMIC_RELAXED, __HIP_MEMORY_SCOPE_AGENT);
    unsigned xcc;
    asm volatile("s_getreg_b32 %0, hwreg(HW_REG_XCC_ID)" : "=s"(xcc));
    xcc &= 7u;
    if (wave == 0) {
      push_role(P, xcc, wi, ctr_v, lane);
      if (lane == 0)
        __hip_atomic_store(ctr, __builtin_amdgcn_readfirstlane(ctr_v) + 1u, __ATOMIC_RELAXED, __HIP_MEMORY_SCOPE_AGENT);
      return;
    }
    const unsigned call = __builtin_amdgcn_readfirstlane(ctr_v) + 1u;
    if (lane == 0) __hip_atomic_store(ctr, call, __ATOMIC_RELAXED, __HIP_MEMORY_SCOPE_AGENT);
    unsigned long long* chk = reinterpret_cast<unsigned long long*>(myws + CHK_OFF) + wi * 32;
    unsigned long long v = 0ull;
    if (lane == 0) v = __hip_atomic_fetch_add(chk + (call & 1u) * 16, (1ull << 32) + (1ull << (4u * xcc)),
                                              __ATOMIC_RELAXED, __HIP_MEMORY_SCOPE_AGENT);
    const unsigned lo = __builtin_amdgcn_readfirstlane((unsigned)v) + (1u << (4u * xcc));
    const unsigned cnt = __builtin_amdgcn_readfirstlane((unsigned)(v >> 32));
    if (cnt == 7u) {
      if (lane == 0) __hip_atomic_store(chk + ((call + 1u) & 1u) * 16, 0ull, __ATOMIC_RELAXED, __HIP_MEMORY_SCOPE_AGENT);
#pragma nounroll
      for (unsigned q = 0; q < 8u; ++q)
        if (((lo >> (4u * q)) & 15u) == 0u) push_role(P, q, wi, ctr_v, lane);
    }
    return;
  }

  unsigned* ctr = reinterpret_cast<unsigned*>(myws + CTR_OFF) + (blockIdx.x * 4 + wave);
  const unsigned ctr_v = __hip_atomic_load(ctr, __ATOMIC_RELAXED, __HIP_MEMORY_SCOPE_AGENT);
  if (threadIdx.x < 4) xflag[threadIdx.x] = 0u;
  __syncthreads();
  const unsigned call = __builtin_amdgcn_readfirstlane(ctr_v) + 1u;
  if (lane == 0) __hip_atomic_store(ctr, call, __ATOMIC_RELAXED, __HIP_MEMORY_SCOPE_AGENT);
  const int gi = wi - 16;
  const int kp = wave;
  const __amdgpu_buffer_rsrc_t rs = make_rsrc(myws);
  const unsigned* fa = reinterpret_cast<const unsigned*>(myws + FLAGA_OFF) + (gi >> 2) * 64 + lane;
  unsigned* tmo = reinterpret_cast<unsigned*>(myws + TMO_OFF);
  const float sc = P.sx[0] * P.sw[0];
  bool failed = false;
  const unsigned long long mine = __ballot((((lane >> 3) + lane) & 3) == kp);
  const unsigned long long t0 = __builtin_amdgcn_s_memrealtime();
  const unsigned abase = AST_OFF + (call & 1u) * NR * A_BYTES + (unsigned)(gi >> 2) * 8192u + (unsigned)(gi & 3) * 2048u;
  unsigned spins = 0;

  for (int ph = 0; ph < 4; ++ph) {
    const int nj = 8 * ph + g;
    const unsigned* fb = reinterpret_cast<const unsigned*>(myws + FLAGB_OFF) + nj * 64 + lane;
    const unsigned bbase = BST_OFF + (call & 1u) * NR * B_BYTES + (unsigned)(ph * 64 + g) * 8192u;
    f32x16 acc[4];
#pragma unroll
    for (int c = 0; c < 4; ++c)
#pragma unroll
      for (int r = 0; r < 16; ++r) acc[c][r] = 0.f;
    unsigned long long done = ~mine;
    unsigned long long ready = 0ull;
    do {
      if (ready == 0ull) {
        do {
          const unsigned va = __hip_atomic_load(fa, __ATOMIC_RELAXED, __HIP_MEMORY_SCOPE_SYSTEM);
          const unsigned vb = __hip_atomic_load(fb, __ATOMIC_RELAXED, __HIP_MEMORY_SCOPE_SYSTEM);
          ready = __ballot(va >= call && vb >= call) & ~done;
          if (ready == 0ull) {
            __builtin_amdgcn_s_sleep(1);
            if ((++spins & 31u) == 0u)
            if (__builtin_amdgcn_s_memrealtime() - t0 > DEADLINE_TICKS ||
                __builtin_amdgcn_readfirstlane(__hip_atomic_load(tmo, __ATOMIC_RELAXED, __HIP_MEMORY_SCOPE_AGENT)) == call) {
              __hip_atomic_store(tmo, call, __ATOMIC_RELAXED, __HIP_MEMORY_SCOPE_AGENT);
              ready = ~done;
              failed = true;
            }
          }
        } while (ready == 0ull);
        asm volatile("" ::: "memory");
        done |= ready;
      }
      consume<UU>(ready, acc, rs, abase, bbase, lane);
    } while ((ready | ~done) != 0ull);
    asm volatile("s_nop 15\n\ts_nop 15" : "+a"(acc[0]), "+a"(acc[1]), "+a"(acc[2]), "+a"(acc[3]));

    const float bad = failed ? __builtin_nanf("") : 0.f;
#pragma unroll
    for (int r = 0; r < 16; ++r)
      xch[ph & 1][wave][r][lane] = f32x4{acc[0][r] + bad, acc[1][r], acc[2][r], acc[3][r]};
    __builtin_amdgcn_fence(__ATOMIC_RELEASE, "workgroup");
    if (lane == 0) __hip_atomic_store(&xflag[wave], (unsigned)(ph + 1), __ATOMIC_RELAXED, __HIP_MEMORY_SCOPE_WORKGROUP);
    while (true) {
      const unsigned f = __hip_atomic_load(&xflag[lane & 3], __ATOMIC_RELAXED, __HIP_MEMORY_SCOPE_WORKGROUP);
      if (__all(f >= (unsigned)(ph + 1))) break;
      __builtin_amdgcn_s_sleep(1);
    }
    __builtin_amdgcn_fence(__ATOMIC_ACQUIRE, "workgroup");
    float* orow = P.out + (size_t)(gi * 32 + kp * 8 + (lane >> 5) * 4) * GN + nj * 128 + (lane & 31) * 4;
#pragma unroll
    for (int r = 0; r < 4; ++r) {
      const f32x4 v = (xch[ph & 1][0][4 * kp + r][lane] + xch[ph & 1][1][4 * kp + r][lane]) +
                      (xch[ph & 1][2][4 * kp + r][lane] + xch[ph & 1][3][4 * kp + r][lane]);
      { f32x4 o = v * sc; asm volatile("global_store_dwordx4 %0, %1, off sc0 sc1\n\ts_nop 1" :: "v"(orow + (size_t)r * GN), "v"(o) : "memory"); }
    }
  }
}

extern "C" void kernel_launch(int rank, int nranks, ncclComm_t comm,
                              hipStream_t stream,
                              void* const* d_in, const int* in_sizes, int n_in,
                              void* d_out, int out_size,
                              const int* in_dtypes, int out_dtype,
                              void* const* d_out_all,
                              void* const* d_ws_all, size_t ws_bytes) {
  if (nranks != NR || ws_bytes < WS_NEEDED || n_in < 4 || out_size != MS * GN ||
      in_sizes[0] != GM * KS || in_sizes[1] != KS * GN) {
    fprintf(stderr, "kernel_launch: unexpected configuration\n");
    return;
  }
  Params P;
  P.x = static_cast<const unsigned char*>(d_in[0]);
  P.w = static_cast<const unsigned char*>(d_in[1]);
  P.sx = static_cast<const float*>(d_in[2]);
  P.sw = static_cast<const float*>(d_in[3]);
  P.out = static_cast<float*>(d_out);
  for (int p = 0; p < NR; ++p) P.ws[p] = static_cast<unsigned char*>(d_ws_all[p]);
  P.rank = rank;
  static const unsigned kMap[NR] = {0x57410362u, 0x62034751u, 0x57416302u, 0x15743602u,
                                    0x60327514u, 0x26307451u, 0x74150623u, 0x20364517u};
  P.map = kMap[rank];
  gemm_ag_fused<<<256, 256, 0, stream>>>(P);
}
```
